```python
import numpy as np

kernel_name = "gpu_a2a_gemm_m2048_k2048_n2048_f32_gelu"


ABSMAX_RTOL = 0.001

COMPUTE_DTYPE = "float32"
EPILOGUE = "gelu"
IN_QUANT = None
OUT_QUANT = None
IN_DTYPES = {"x": "float32", "w_mat": "float32"}
WORLD_SIZE = 8

def setup_inputs():

    x = np.random.randn(2048, 2048).astype(np.float32)
    w_mat = (np.random.randn(2048, 2048) / np.sqrt(2048)).astype(np.float32)
    return {"x": x, "w_mat": w_mat}

def _epilogue(y):
    c = 0.7978845608028654
    return (0.5 * y * (1.0 + np.tanh(c * (y + 0.044715 * y ** 3)))).astype(np.float32)

def reference(x, w_mat):
    y = (x @ w_mat).astype(np.float32)
    return _epilogue(y)
```

```cpp
#include <hip/hip_runtime.h>
#include <rccl/rccl.h>
#include <cstdio>

#define HIP_CHECK(expr) do { hipError_t _e = (expr); if (_e != hipSuccess) { \
    fprintf(stderr, "HIP error %s at %s:%d\n", hipGetErrorString(_e), __FILE__, __LINE__); \
    return; } } while (0)

typedef unsigned u32;
typedef unsigned u32x2 __attribute__((ext_vector_type(2)));
typedef unsigned u32x4 __attribute__((ext_vector_type(4)));
typedef float f32x2 __attribute__((ext_vector_type(2)));
typedef float f32x4 __attribute__((ext_vector_type(4)));
typedef _Float16 f16x2 __attribute__((ext_vector_type(2)));
typedef _Float16 f16x8 __attribute__((ext_vector_type(8)));

constexpr int NR = 8;
constexpr int MG = 2048, KG = 2048, NG = 2048;
constexpr int MS = MG / NR;
constexpr int KS = KG / NR;
constexpr u32 CNT_OFF  = 0;
constexpr u32 TMO_OFF  = 32768;
constexpr u32 FLAG_OFF = 65536;
constexpr u32 DATA_OFF = 1u << 20;
constexpr u32 BUF_BYTES = 1u << 20;
constexpr u32 WS_USED  = DATA_OFF + 2 * BUF_BYTES;
constexpr unsigned long long DEADLINE_TICKS = 40000000ull;

struct Params {
    const float* x;
    const float* w;
    float* out;
    char* ws[NR];
    int rank;
    int pad;
};

__device__ __forceinline__ u32 pk(float a, float b) {
    f32x2 v = {a, b};
    return __builtin_bit_cast(u32, __builtin_convertvector(v, f16x2));
}
__device__ __forceinline__ float gelu(float y) {
    const float c = 0.7978845608028654f;
    float u = c * (y + 0.044715f * y * y * y);
    float e = __builtin_amdgcn_exp2f(u * -2.8853900817779268f);
    return y * __builtin_amdgcn_rcpf(1.0f + e);
}

__global__ void __launch_bounds__(512, 2) a2a_gemm(Params p) {
    __shared__ __attribute__((aligned(16))) char smem[8 * 8 * 1280];
    const int b = blockIdx.x;
    const int lane = threadIdx.x & 63;
    const int wave = __builtin_amdgcn_readfirstlane(threadIdx.x >> 6);
    const int rank = p.rank;
    char* myws = p.ws[rank & 7];
    const int g = lane >> 4, r16 = lane & 15;

    u32* cnt = (u32*)(myws + CNT_OFF + b * 128);
    const u32 call = __builtin_amdgcn_readfirstlane(
        __hip_atomic_load(cnt, __ATOMIC_RELAXED, __HIP_MEMORY_SCOPE_AGENT)) + 1;
    const u32 buf = call & 1;

    {
        const int dest = (rank + 1 + (b & 7)) & 7;
        const int mt = (b >> 3) & 3, kb = b >> 5;
        if (wave < 4) {
            const float* src = p.x + (size_t)(dest * MS + mt * 64 + wave * 16 + r16) * KS + kb * 32 + g * 8;
            f32x4 v0 = *(const f32x4*)src, v1 = *(const f32x4*)(src + 4);
            u32x4 h = {pk(v0[0], v0[1]), pk(v0[2], v0[3]), pk(v1[0], v1[1]), pk(v1[2], v1[3])};
            char* dst = p.ws[dest] + DATA_OFF + buf * BUF_BYTES + ((rank * 4 + mt) * 8 + kb) * 4096 + wave * 1024 + lane * 16;
            asm volatile("global_store_dwordx4 %0, %1, off sc0 sc1\n\ts_nop 1" :: "v"(dst), "v"(h) : "memory");
            asm volatile("s_waitcnt vmcnt(0)" ::: "memory");
        }
        __syncthreads();
        if (threadIdx.x == 0) {
            u32* flag = (u32*)(p.ws[dest] + FLAG_OFF + (rank * 4 + mt) * 128 + kb * 16);
            __hip_atomic_store(flag, call, __ATOMIC_RELAXED, __HIP_MEMORY_SCOPE_SYSTEM);
        }
    }

    const int ntile = (b >> 3) & 7, mtile = b >> 6;
    const int n0 = (b & 7) * 256 + ntile * 32;
    const int s = wave;
    u32x4 wf[8][2];
    {
        auto rw = __builtin_amdgcn_make_buffer_rsrc((void*)p.w, 0, (u32)KG * NG * 4, 0x00020000);
        const u32 voff = (u32)(g * 8 * NG + n0 + 2 * r16) * 4;
        const u32 sbase = (u32)s * KS * NG * 4;
        f32x2 t[8][8];
#pragma unroll
        for (int kb = 0; kb < 8; ++kb)
#pragma unroll
            for (int j = 0; j < 8; ++j)
                t[kb][j] = __builtin_bit_cast(f32x2, __builtin_amdgcn_raw_buffer_load_b64(rw, voff, sbase + (kb * 32 + j) * NG * 4, 0));
#pragma unroll
        for (int kb = 0; kb < 8; ++kb)
#pragma unroll
            for (int c = 0; c < 2; ++c)
#pragma unroll
                for (int j = 0; j < 4; ++j)
                    wf[kb][c][j] = pk(t[kb][2 * j][c], t[kb][2 * j + 1][c]);
    }

#pragma unroll
    for (int kb = 0; kb < 8; ++kb) { asm volatile("" : "+v"(wf[kb][0])); asm volatile("" : "+v"(wf[kb][1])); }
    bool bad = false;
    {
        u32* flag = (u32*)(myws + FLAG_OFF + (s * 4 + mtile) * 128 + (lane & 7) * 16);
        unsigned long long t0 = __builtin_amdgcn_s_memrealtime();
        while (true) {
            u32 f = __hip_atomic_load(flag, __ATOMIC_RELAXED, __HIP_MEMORY_SCOPE_SYSTEM);
            if (__all((int)(f - call) >= 0)) break;
            if (__builtin_amdgcn_s_memrealtime() - t0 > DEADLINE_TICKS) { bad = true; break; }
            __builtin_amdgcn_s_sleep(1);
        }
        asm volatile("" ::: "memory");
    }

    f32x4 acc[4][2];
#pragma unroll
    for (int rb = 0; rb < 4; ++rb) { acc[rb][0] = f32x4{0, 0, 0, 0}; acc[rb][1] = f32x4{0, 0, 0, 0}; }
    {
        auto ra = __builtin_amdgcn_make_buffer_rsrc((void*)(myws + DATA_OFF), 0, 2 * BUF_BYTES, 0x00020000);
        const u32 sbase = buf * BUF_BYTES + (s * 4 + mtile) * 32768;
        u32x4 a[8][4];
#pragma unroll
        for (int kb = 0; kb < 8; ++kb)
#pragma unroll
            for (int rb = 0; rb < 4; ++rb)
                a[kb][rb] = __builtin_amdgcn_raw_buffer_load_b128(ra, lane * 16 + rb * 1024, sbase + kb * 4096, 17);
        __builtin_amdgcn_sched_barrier(0);
#pragma unroll
        for (int kb = 0; kb < 8; ++kb)
#pragma unroll
            for (int rb = 0; rb < 4; ++rb)
#pragma unroll
                for (int c = 0; c < 2; ++c)
                    acc[rb][c] = __builtin_amdgcn_mfma_f32_16x16x32_f16(
                        __builtin_bit_cast(f16x8, wf[kb][c]), __builtin_bit_cast(f16x8, a[kb][rb]), acc[rb][c], 0, 0, 0);
    }
    if (bad) {
        const float nan = __builtin_nanf("");
#pragma unroll
        for (int rb = 0; rb < 4; ++rb) { acc[rb][0] = f32x4{nan, nan, nan, nan}; acc[rb][1] = f32x4{nan, nan, nan, nan}; }
    }

    {
        char* wbase = smem + wave * 8 * 1280 + lane * 16 + g * 64;
#pragma unroll
        for (int rb = 0; rb < 4; ++rb)
#pragma unroll
            for (int c = 0; c < 2; ++c)
                *(f32x4*)(wbase + (rb * 2 + c) * 1280) = acc[rb][c];
    }
    __syncthreads();
    {
        const int rb = wave >> 1, row = (wave & 1) * 8 + (lane >> 3), cq = lane & 7;
        const char* rbase = smem + rb * 2 * 1280 + (cq >> 1) * 320 + row * 16 + (cq & 1) * 8;
        f32x2 s0 = {0, 0}, s1 = {0, 0};
#pragma unroll
        for (int w = 0; w < 8; ++w) {
            s0 += *(const f32x2*)(rbase + w * 8 * 1280);
            s1 += *(const f32x2*)(rbase + w * 8 * 1280 + 1280);
        }
        f32x4 o = {gelu(s0[0]), gelu(s1[0]), gelu(s0[1]), gelu(s1[1])};
        float* dst = p.out + (size_t)(mtile * 64 + rb * 16 + row) * NG + n0 + cq * 4;
        *(f32x4*)dst = o;
    }
    if (threadIdx.x == 0) __hip_atomic_store(cnt, call, __ATOMIC_RELAXED, __HIP_MEMORY_SCOPE_AGENT);
}

extern "C" void kernel_launch(int rank, int nranks, ncclComm_t comm,
                              hipStream_t stream,
                              void* const* d_in, const int* in_sizes, int n_in,
                              void* d_out, int out_size,
                              const int* in_dtypes, int out_dtype,
                              void* const* d_out_all,
                              void* const* d_ws_all, size_t ws_bytes) {
    if (nranks != NR || (unsigned)rank >= (unsigned)NR || n_in < 2 || ws_bytes < WS_USED ||
        in_sizes[0] != MG * KS || in_sizes[1] != KG * NG || out_size != MS * NG) {
        fprintf(stderr, "kernel_launch: unexpected problem shape\n");
        return;
    }
    Params p;
    p.x = (const float*)d_in[0];
    p.w = (const float*)d_in[1];
    p.out = (float*)d_out;
    for (int i = 0; i < NR; ++i) p.ws[i] = (char*)d_ws_all[i];
    p.rank = rank;
    p.pad = 0;
    a2a_gemm<<<dim3(256), dim3(512), 0, stream>>>(p);
    HIP_CHECK(hipGetLastError());
}
```

```cpp
#include <hip/hip_runtime.h>
#include <rccl/rccl.h>
#include <cstdio>

#define HIP_CHECK(expr) do { hipError_t _e = (expr); if (_e != hipSuccess) { \
    fprintf(stderr, "HIP error %s at %s:%d\n", hipGetErrorString(_e), __FILE__, __LINE__); \
    return; } } while (0)

typedef unsigned u32;
typedef unsigned long long u64;
typedef unsigned u32x2 __attribute__((ext_vector_type(2)));
typedef unsigned u32x4 __attribute__((ext_vector_type(4)));
typedef float f32x2 __attribute__((ext_vector_type(2)));
typedef float f32x4 __attribute__((ext_vector_type(4)));
typedef _Float16 f16x2 __attribute__((ext_vector_type(2)));
typedef _Float16 f16x8 __attribute__((ext_vector_type(8)));
#define LDSP __attribute__((address_space(3)))

#define PACE0 2400
#define PACED 145
#define OUT_AUX 0
#define XCH_AUX 16
#define ZERO_AUX 17
#define GATE_G 3
#define ASLIDE 1
#define MAP 0
#define WARM 0
#define WARMP 0
#define CNT_AUX 16
#define XFIRST 1
#define PFD 4
#define XK 2
#define SGMAP 1
#define ROT 1

constexpr int NR = 8;
constexpr int MG = 2048, KG = 2048, NG = 2048;
constexpr int MS = MG / NR, KS = KG / NR;
constexpr u32 CNT_OFF  = 0;
constexpr u32 DATA_OFF = 1u << 20;
constexpr u32 BUF_BYTES = 1u << 20;
constexpr u32 XCH_OFF  = 4u << 20;
constexpr u32 WS_USED  = 8u << 20;
constexpr u64 DEADLINE_TICKS = 40000000ull;
constexpr int FS = 1088, KBS = 2 * FS;
constexpr int LDS_W = 32 * KBS;
constexpr int LDS_X = LDS_W;
constexpr int LDS_S = LDS_X + 4 * 1088;
constexpr int LDS_BYTES = 84 * 1024;

struct M0 { static constexpr int value = 0; };
struct M1 { static constexpr int value = 1; };
struct M2 { static constexpr int value = 2; };
struct Params {
    const float* x;
    const float* w;
    float* out;
    char* myws;
    char* ws[NR];
    int rank;
    int pad;
};

__device__ __forceinline__ u32 pk(float a, float b) {
    f32x2 v = {a, b};
    return __builtin_bit_cast(u32, __builtin_convertvector(v, f16x2));
}
__device__ __forceinline__ u32 pknz(float a, float b) {
    const u32 d = pk(a, b);
    return d == 0u ? 0x80000000u : d;
}
__device__ __forceinline__ float gelu(float y) {
    const float c = 0.7978845608028654f;
    float u = c * (y + 0.044715f * y * y * y);
    float e = __builtin_amdgcn_exp2f(u * -2.8853900817779268f);
    return y * __builtin_amdgcn_rcpf(1.0f + e);
}
__device__ __forceinline__ u32 min3u(u32 a, u32 b, u32 c) { return min(min(a, b), c); }
#define SB() __builtin_amdgcn_sched_barrier(0)
#define KEEP(x) asm volatile("" :: "v"(x))
#define CYC(v) do {} while (0)
#define STAMP(i) do {} while (0)
#define STAMP_OUT() do {} while (0)

__global__ void __launch_bounds__(576, 3) a2a_gemm(Params p) {
    __shared__ __attribute__((aligned(16))) char smem[LDS_BYTES];
    STAMP(0);
    CYC(cy0);
    const int b = blockIdx.x;
    const int lane = threadIdx.x & 63;
    const int wave = __builtin_amdgcn_readfirstlane(threadIdx.x >> 6);
    const int di = b & 7;
    int rank = p.rank;
    char* dws = p.ws[di];
    const float* xp = p.x; const float* wp_ = p.w; float* outp = p.out; char* myws = p.myws;
    LDSP char* lds = (LDSP char*)smem;
    volatile LDSP u32* sync = (volatile LDSP u32*)(lds + LDS_S);

    if (__builtin_expect(wave == 0, 1)) {
        __builtin_amdgcn_s_setprio(3);
        const u64 tstart = __builtin_amdgcn_s_memtime();
        const int sg = (b >> 4) | (((b >> 3) & 1) << 4);
        u32 xv = (u32)((lane >> 5) * KS + (lane & 31) * 4) * 4 + (u32)((((sg >> 3) & 1) * 8 + ((sg >> 1) & 3)) * 16 * KS + (sg & 1) * 128) * 4;
        sync[lane & 15] = 0;
        asm volatile("" : "+v"(xv));
        asm volatile("" : "+s"(rank), "+s"(dws), "+s"(xp), "+s"(myws));
        const int dest = (rank + 1 + di) & 7;
        const int half = ((sg >> 4) ^ 1 ^ (rank >> 2)) & 1;
        auto rx = __builtin_amdgcn_make_buffer_rsrc((void*)xp, 0, MG * KS * 4, 0x00020000);
        auto rws = __builtin_amdgcn_make_buffer_rsrc((void*)myws, 0, WS_USED, 0x00020000);
        const u32 xs = (u32)((dest * MS + half * 64) * KS) * 4;
        f32x4 v[8];
        STAMP(5);
#pragma unroll
        for (int j = 0; j < 8; ++j) v[j] = __builtin_bit_cast(f32x4, __builtin_amdgcn_raw_buffer_load_b128(rx, xv + (j & 1) * 2048, xs + (j >> 1) * 4096, 0));
        u32 cv = __builtin_amdgcn_raw_buffer_load_b32(rws, 0, CNT_OFF + b * 128, 16);
        SB();
        STAMP(6);
        __builtin_amdgcn_s_barrier();
        SB();
        STAMP(1);
        const int k4 = lane & 31;
        LDSP char* xw = lds + LDS_X + (k4 >> 3) * 1088 + ((k4 >> 1) & 3) * 272 + (lane >> 5) * 16 + (k4 & 1) * 8;
        LDSP char* xr = lds + LDS_X + (lane >> 4) * 272 + (lane & 15) * 16;
        const int rb16 = ((sg >> 3) & 1) * 8 + half * 4 + ((sg >> 1) & 3);
        auto rd = __builtin_amdgcn_make_buffer_rsrc((void*)dws, 0, WS_USED, 0x00020000);
        u32 dv = lane * 16 + DATA_OFF + ((rank * 16 + rb16) * 8 + (sg & 1) * 4) * 1024;
        u32 zv = lane * 16 + DATA_OFF + b * 4096;
        asm volatile("" : "+v"(dv), "+v"(zv), "+v"(xw), "+v"(xr));
        SB();
#pragma unroll
        for (int j = 0; j < 8; ++j) *(LDSP u32x2*)(xw + j * 32) = u32x2{pk(v[j][0], v[j][1]), pk(v[j][2], v[j][3])};
        STAMP(7);
        u32x4 hh[4];
#pragma unroll
        for (int f = 0; f < 4; ++f) hh[f] = *(LDSP u32x4*)(xr + f * 1088);
        SB();
        asm volatile("" : "+v"(cv));
        const u32 call = __builtin_amdgcn_readfirstlane(cv) + 1;
        const u32 c3 = call % 3u;
        const u32 z3 = c3 == 0 ? 2u : c3 - 1;
        sync[4] = call;
        {
            u32 mn = 0xFFFFFFFFu;
#pragma unroll
            for (int f = 0; f < 4; ++f) mn = min3u(min3u(hh[f][0], hh[f][1], hh[f][2]), hh[f][3], mn);
            if (__builtin_expect(__builtin_amdgcn_ballot_w64(mn == 0) != 0, 0)) {
#pragma unroll
                for (int f = 0; f < 4; ++f)
#pragma unroll
                    for (int e = 0; e < 4; ++e) hh[f][e] |= (hh[f][e] - 1u) & 0x80000000u;
            }
        }
        dv += c3 * BUF_BYTES; zv += z3 * BUF_BYTES;
        u32x4 zz = {0, 0, 0, 0};
        asm volatile("" : "+v"(hh[0]), "+v"(hh[1]), "+v"(hh[2]), "+v"(hh[3]), "+v"(dv), "+v"(zv), "+v"(zz));
        STAMP(2);
        if (di != 7) {
            const u64 tgt = tstart + PACE0 + sg * PACED;
            while (__builtin_amdgcn_s_memtime() < tgt) {}
        }
        SB();
#pragma unroll
        for (int f = 0; f < 4; ++f) __builtin_amdgcn_raw_buffer_store_b128(hh[f], rd, dv + f * 1024, 0, 17);
#pragma unroll
        for (int f = 0; f < 4; ++f) __builtin_amdgcn_raw_buffer_store_b128(zz, rws, zv + f * 1024, 0, ZERO_AUX);
        __builtin_amdgcn_raw_buffer_store_b32(call, rws, 0, CNT_OFF + b * 128, CNT_AUX);
        STAMP(3);
        STAMP_OUT();
        __builtin_amdgcn_endpgm();
    }

    const int cw = wave - 1;
    const bool late = cw >= 4;
    const int i5 = b >> 3;
    const int mh = i5 & 1, kh = (i5 >> 1) & 1, ntile = i5 >> 2;
    const int n0 = (b & 7) * 256 + ntile * 32;
    const int partner = b ^ 16;
    const int g = lane >> 4, h = (lane >> 3) & 1, c = lane & 7, r16 = lane & 15;
    u32 wv = (u32)((8 * g + 4 * h) * NG + n0 + 4 * c) * 4 + (u32)(kh * 1024 + cw * 32) * NG * 4;
    asm volatile("" : "+v"(wv));
    asm volatile("" : "+s"(wp_), "+s"(outp), "+s"(myws));
    const u32 lidx = ((u32)(size_t)wp_ >> 7) + (n0 >> 5);
    const int wrot = (((lidx & 7) == 3 || (lidx & 63) == 63) && mh) ? 2 : 0;
    auto rw = __builtin_amdgcn_make_buffer_rsrc((void*)wp_, 0, (u32)KG * NG * 4, 0x00020000);
    f32x4 t[2][4];
#define WLOAD(pw, j) __builtin_bit_cast(f32x4, __builtin_amdgcn_raw_buffer_load_b128(rw, wv, ((((pw) ^ wrot) * 256 + (j)) * NG * 4), 0))
    SB();
    __builtin_amdgcn_s_barrier();
    SB();
#pragma unroll
    for (int j = 0; j < 4; ++j) t[0][j] = WLOAD(0, j);
    auto rws = __builtin_amdgcn_make_buffer_rsrc((void*)myws, 0, WS_USED, 0x00020000);
    SB();
    STAMP(1);
    const int rb16 = mh * 8 + (late ? kh : 1 - kh) * 4 + (cw & 3);
    LDSP char* ww = lds + cw * KBS + (c >> 2) * FS + g * 256 + (c & 3) * 16 + h * 8;
    auto wstore = [&](int pw) {
        LDSP char* q = ww + (pw ^ wrot) * (8 * KBS);
#pragma unroll
        for (int u = 0; u < 4; ++u)
            *(LDSP u32x2*)(q + u * 64) = u32x2{pk(t[pw & 1][0][u], t[pw & 1][1][u]), pk(t[pw & 1][2][u], t[pw & 1][3][u])};
    };
#pragma unroll
    for (int pw = 0; pw < 3; ++pw) {
#pragma unroll
        for (int j = 0; j < 4; ++j) { KEEP(t[pw & 1][j]); SB(); t[(pw + 1) & 1][j] = WLOAD(pw + 1, j); SB(); }
        wstore(pw);
        SB();
        STAMP(2 + pw);
    }
    u32 call;
    do { call = __builtin_amdgcn_readfirstlane(sync[4]); } while (call == 0);
    bool bad = false;
    const u32 abase = DATA_OFF + (call % 3u) * BUF_BYTES + (kh * 4 * 16 + rb16) * 8192;
    u32 av = lane * 16, av2 = lane * 16 + 4096;
    asm volatile("" : "+v"(av), "+v"(av2));
    const int rot = ROT ? ntile : 0;
    u32x4 a[8], wf[PFD][2];
    f32x4 acc[4][2];
    LDSP char* wr = lds + g * 256 + ((r16 >> 2) + 4 * (r16 & 3)) * 16;
#define ALOAD(k, so) __builtin_amdgcn_raw_buffer_load_b128(rws, ((k) < 4 ? av : av2) + ((k) & 3) * 1024, so, 17)
#define MFMA(w, x, c) __builtin_amdgcn_mfma_f32_16x16x32_f16(__builtin_bit_cast(f16x8, w), __builtin_bit_cast(f16x8, x), c, 0, 0, 0)
    auto aoff = [&](int G) { return abase + ((G + rot) & 3) * (16 * 8192); };
    auto wptr = [&](int G) { return wr + ((G + rot) & 3) * (8 * KBS); };
    auto issue = [&](int G) {
        const u32 so = aoff(G);
#pragma unroll
        for (int k = 0; k < 8; ++k) { a[k] = ALOAD(k, so); SB(); }
    };
    LDSP char* wq[4] = {wptr(0), wptr(1), wptr(2), wptr(3)};
    asm volatile("" : "+v"(wq[0]), "+v"(wq[1]), "+v"(wq[2]), "+v"(wq[3]));
    auto wpre = [&]() {
#pragma unroll
        for (int k = 0; k < PFD; ++k) { wf[k][0] = *(LDSP u32x4*)(wq[0] + k * KBS); wf[k][1] = *(LDSP u32x4*)(wq[0] + k * KBS + FS); }
        SB();
    };
    volatile LDSP u32* gate = sync + (cw & 3);
    volatile LDSP u32* wrdy = sync + 4 + min(lane, 8);
    auto finish_w = [&]() {
        asm volatile("" : "+v"(t[1][0]), "+v"(t[1][1]), "+v"(t[1][2]), "+v"(t[1][3]));
        wstore(3);
        sync[4 + wave] = 1;
        SB();
        STAMP(5);
        while (__builtin_amdgcn_ballot_w64(*wrdy == 0) != 0) {}
        asm volatile("" ::: "memory");
        SB();
        wpre();
    };
    const u32 xo_in = XCH_OFF + (b * 4 + (cw & 3)) * 4096;
    u32x4 y[4];
    u32 fail = 0;

    auto body = [&](auto mode, int G) {
        constexpr int MODE = decltype(mode)::value;
        const u32 so = aoff(G + 1);
        u32 mn = 0xFFFFFFFFu, mn2 = 0xFFFFFFFFu;
        f32x4 c0 = {0, 0, 0, 0}, c1 = {0, 0, 0, 0};
#pragma unroll
        for (int k = 0; k < 8; ++k) {
            const bool wl = k + PFD < 8 || MODE == 0;
            LDSP char* wn = k + PFD < 8 ? wq[G] + (k + PFD) * KBS : wq[(G + 1) & 3] + (k + PFD - 8) * KBS;
            asm volatile("" :: "v"(wf[k % PFD][0]), "v"(wf[k % PFD][1]), "v"(a[k]));
            SB();
            c0 = MFMA(wf[k % PFD][0], a[k], c0);
            SB();
            mn = min3u(a[k][0], a[k][1], mn);
            KEEP(mn);
            SB();
            if (wl) wf[k % PFD][0] = *(LDSP u32x4*)wn;
            SB();
            c1 = MFMA(wf[k % PFD][1], a[k], c1);
            SB();
            mn2 = min3u(a[k][2], a[k][3], mn2);
            KEEP(mn2);
            SB();
            if (wl) wf[k % PFD][1] = *(LDSP u32x4*)(wn + FS);
            SB();
            if (MODE == 0) a[k] = ALOAD(k, so);
            if (MODE == 2 && k >= XK && k < XK + 4) y[k - XK] = __builtin_amdgcn_raw_buffer_load_b128(rws, av + (k - XK) * 1024, xo_in, 16);
            SB();
        }
        mn = min(mn, mn2);
        acc[G][0] = c0; acc[G][1] = c1;
        const u64 bm = __builtin_amdgcn_ballot_w64(mn == 0);
        fail |= min((u32)bm | (u32)(bm >> 32), 1u) << G;
        if (G == GATE_G - 1 && !late) *gate = 1;
        SB();
    };
    f32x4 s0, s1;
    auto combine = [&]() {
        if (__builtin_expect(fail == 0, 1)) {
            s0 = (acc[0][0] + acc[1][0]) + (acc[2][0] + acc[3][0]);
            s1 = (acc[0][1] + acc[1][1]) + (acc[2][1] + acc[3][1]);
        } else {
            s0 = f32x4{0, 0, 0, 0}; s1 = f32x4{0, 0, 0, 0};
#pragma unroll
            for (int G = 0; G < 4; ++G) if (!(fail >> G & 1)) { s0 += acc[G][0]; s1 += acc[G][1]; }
            const u64 t0 = __builtin_amdgcn_s_memrealtime();
            int G = 0;
#pragma nounroll
            while (fail != 0) {
                G = (G + __builtin_ctz(((fail | (fail << 4)) >> G))) & 3;
                asm volatile("" ::: "memory");
                issue(G);
                LDSP char* wp = wptr(G);
                u32x4 w0[4], w1[4];
#pragma unroll
                for (int k = 0; k < 4; ++k) { w0[k] = *(LDSP u32x4*)(wp + k * KBS); w1[k] = *(LDSP u32x4*)(wp + k * KBS + FS); }
                SB();
                f32x4 c0 = s0, c1 = s1;
                u32 mn = 0xFFFFFFFFu;
#pragma unroll
                for (int k = 0; k < 8; ++k) {
                    c0 = MFMA(w0[k & 3], a[k], c0);
                    c1 = MFMA(w1[k & 3], a[k], c1);
                    mn = min3u(min3u(a[k][0], a[k][1], a[k][2]), a[k][3], mn);
                    SB();
                    if (k < 4) { w0[k] = *(LDSP u32x4*)(wp + (k + 4) * KBS); w1[k] = *(LDSP u32x4*)(wp + (k + 4) * KBS + FS); }
                    SB();
                }
                if (__builtin_amdgcn_ballot_w64(mn == 0) == 0) { s0 = c0; s1 = c1; fail &= ~(1u << G); }
                else if (__builtin_amdgcn_s_memrealtime() - t0 > DEADLINE_TICKS) { bad = true; fail = 0; }
                G = (G + 1) & 3;
            }
        }
    };
    if (!late) {
        __builtin_amdgcn_s_setprio(2);
        {
            const u32 so = aoff(0);
#pragma unroll
            for (int j = 0; j < 4; ++j) { KEEP(t[1][j]); SB(); a[2 * j] = ALOAD(2 * j, so); a[2 * j + 1] = ALOAD(2 * j + 1, so); SB(); }
        }
        finish_w();
        STAMP(6);
        CYC(cy1);
        body(M0{}, 0);
        STAMP(7);
        body(M0{}, 1);
        STAMP(8);
        body(M0{}, 2);
        STAMP(9);
        body(M1{}, 3);
        combine();
        if (bad) { const float nan = __builtin_nanf(""); s0 = f32x4{nan, nan, nan, nan}; }
        u32x4 x0 = __builtin_bit_cast(u32x4, s0), x1 = __builtin_bit_cast(u32x4, s1);
        const u32 xo = XCH_OFF + (partner * 4 + cw) * 4096;
        STAMP(10);
        u32x4 e0 = {x0[0], call, x0[1], call}, e1 = {x0[2], call, x0[3], call}, e2 = {x1[0], call, x1[1], call}, e3 = {x1[2], call, x1[3], call};
        u32 xa = av + xo;
        asm volatile("" : "+v"(e0), "+v"(e1), "+v"(e2), "+v"(e3), "+v"(xa));
        __builtin_amdgcn_raw_buffer_store_b128(e0, rws, xa, 0, XCH_AUX);
        __builtin_amdgcn_raw_buffer_store_b128(e1, rws, xa + 1024, 0, XCH_AUX);
        __builtin_amdgcn_raw_buffer_store_b128(e2, rws, xa + 2048, 0, XCH_AUX);
        __builtin_amdgcn_raw_buffer_store_b128(e3, rws, xa + 3072, 0, XCH_AUX);
        STAMP_OUT();
        __builtin_amdgcn_endpgm();
    }
    {
        u32 ov = (u32)((rb16 * 16 + r16) * NG + n0 + g * 4) * 4;
        asm volatile("" : "+v"(ov));
        finish_w();
        while (__builtin_amdgcn_readfirstlane(*gate) == 0) __builtin_amdgcn_s_sleep(1);
        asm volatile("" ::: "memory");
        issue(0);
        STAMP(6);
        CYC(cy1);
        body(M0{}, 0);
        STAMP(7);
        body(M0{}, 1);
        STAMP(8);
        body(M0{}, 2);
        STAMP(9);
        body(M2{}, 3);
        combine();
        STAMP(13);
        auto tagbad = [&]() {
            u32 d = 0;
#pragma unroll
            for (int k = 0; k < 4; ++k) d |= (y[k][1] ^ call) | (y[k][3] ^ call);
            return __builtin_amdgcn_ballot_w64(d != 0) != 0;
        };
        if (__builtin_expect(tagbad(), 0)) {
            const u64 t0 = __builtin_amdgcn_s_memrealtime();
            do {
                asm volatile("" ::: "memory");
#pragma unroll
                for (int k = 0; k < 4; ++k) y[k] = __builtin_amdgcn_raw_buffer_load_b128(rws, av + k * 1024, xo_in, 16);
                if (__builtin_amdgcn_s_memrealtime() - t0 > DEADLINE_TICKS) bad = true;
            } while (tagbad() && !bad);
        }
        STAMP(14);
        f32x4 p0 = __builtin_bit_cast(f32x4, u32x4{y[0][0], y[0][2], y[1][0], y[1][2]});
        f32x4 p1 = __builtin_bit_cast(f32x4, u32x4{y[2][0], y[2][2], y[3][0], y[3][2]});
        s0 += p0; s1 += p1;
        if (bad) { const float nan = __builtin_nanf(""); s0 = f32x4{nan, nan, nan, nan}; }
        f32x4 o0 = {gelu(s0[0]), gelu(s0[1]), gelu(s0[2]), gelu(s0[3])};
        f32x4 o1 = {gelu(s1[0]), gelu(s1[1]), gelu(s1[2]), gelu(s1[3])};
        auto ro = __builtin_amdgcn_make_buffer_rsrc((void*)outp, 0, MS * NG * 4, 0x00020000);
        __builtin_amdgcn_raw_buffer_store_b128(__builtin_bit_cast(u32x4, o0), ro, ov, 0, OUT_AUX);
        __builtin_amdgcn_raw_buffer_store_b128(__builtin_bit_cast(u32x4, o1), ro, ov + 64, 0, OUT_AUX);
        STAMP_OUT();
    }
}

extern "C" void kernel_launch(int rank, int nranks, ncclComm_t comm,
                              hipStream_t stream,
                              void* const* d_in, const int* in_sizes, int n_in,
                              void* d_out, int out_size,
                              const int* in_dtypes, int out_dtype,
                              void* const* d_out_all,
                              void* const* d_ws_all, size_t ws_bytes) {
    if (nranks != NR || (unsigned)rank >= (unsigned)NR || n_in < 2 || ws_bytes < WS_USED ||
        in_sizes[0] != MG * KS || in_sizes[1] != KG * NG || out_size != MS * NG) {
        fprintf(stderr, "kernel_launch: unexpected problem shape\n");
        return;
    }
    Params p;
    p.x = (const float*)d_in[0];
    p.w = (const float*)d_in[1];
    p.out = (float*)d_out;
    p.myws = (char*)d_ws_all[rank];
    for (int i = 0; i < NR; ++i) p.ws[i] = (char*)d_ws_all[(rank + 1 + i) & 7];
    p.rank = rank;
    p.pad = 0;
    a2a_gemm<<<dim3(256), dim3(576), 0, stream>>>(p);
    HIP_CHECK(hipGetLastError());
}
```

```cpp
#include <hip/hip_runtime.h>
#include <rccl/rccl.h>
#include <cstdio>

#define HIP_CHECK(expr) do { hipError_t _e = (expr); if (_e != hipSuccess) { \
    fprintf(stderr, "HIP error %s at %s:%d\n", hipGetErrorString(_e), __FILE__, __LINE__); \
    return; } } while (0)

typedef unsigned u32;
typedef unsigned long long u64;
typedef unsigned u32x2 __attribute__((ext_vector_type(2)));
typedef unsigned u32x4 __attribute__((ext_vector_type(4)));
typedef unsigned u32x8 __attribute__((ext_vector_type(8)));
typedef float f32x2 __attribute__((ext_vector_type(2)));
typedef float f32x4 __attribute__((ext_vector_type(4)));
typedef _Float16 f16x2 __attribute__((ext_vector_type(2)));
typedef _Float16 f16x8 __attribute__((ext_vector_type(8)));
#define LDSP __attribute__((address_space(3)))

constexpr int NR = 8;
constexpr int MG = 2048, KG = 2048, NG = 2048;
constexpr int MS = MG / NR, KS = KG / NR;
constexpr u32 CNT_OFF   = 0;
constexpr u32 DATA_OFF  = 1u << 20;
constexpr u32 BUF_BYTES = 1u << 20;
constexpr u32 XCH_OFF   = 4u << 20;
constexpr u32 WS_USED   = 8u << 20;
constexpr u32 DEADLINE_TICKS = 40000000u;
constexpr int LDS_P = 0;
constexpr int LDS_X = 65536;
constexpr int LDS_S = LDS_X + 4 * 1120;
constexpr int LDS_BYTES = LDS_S + 64;

struct Params {
    const float* x;
    const float* w;
    float* out;
    char* myws;
    char* ws[NR];
    u64 pad;
    u32x2 cfg[256];
};
static_assert(__builtin_offsetof(Params, ws) == 0x20 && __builtin_offsetof(Params, cfg) == 0x68, "offsets used by the argument loads");

__device__ __forceinline__ u32 pk(float a, float b) {
    f32x2 v = {a, b};
    return __builtin_bit_cast(u32, __builtin_convertvector(v, f16x2));
}
__device__ __forceinline__ f32x4 gelu4(f32x4 y) {
    const float k1 = -2.8853900817779268f * 0.7978845608028654f, k2 = k1 * 0.044715f;
    const f32x4 a = y * __builtin_elementwise_fma(y * y, f32x4{k2, k2, k2, k2}, f32x4{k1, k1, k1, k1});
    const f32x4 d = f32x4{__builtin_amdgcn_exp2f(a[0]), __builtin_amdgcn_exp2f(a[1]), __builtin_amdgcn_exp2f(a[2]), __builtin_amdgcn_exp2f(a[3])} + 1.0f;
    return y * f32x4{__builtin_amdgcn_rcpf(d[0]), __builtin_amdgcn_rcpf(d[1]), __builtin_amdgcn_rcpf(d[2]), __builtin_amdgcn_rcpf(d[3])};
}
__device__ __forceinline__ u32 min3u(u32 a, u32 b, u32 c) { return min(min(a, b), c); }
__device__ __forceinline__ u32x4 mkrsrc(const void* p, u32 bytes) {
    const u64 a = (u64)p;
    return u32x4{(u32)a, (u32)(a >> 32) & 0xffffu, bytes, 0x00020000u};
}
#define SB() __builtin_amdgcn_sched_barrier(0)
#define KEEP(x) asm volatile("" :: "v"(x))
#define NZ(z) ((z) | (((z) - 1u) & 0x80000000u))
#define MFMA(x, w, c) __builtin_amdgcn_mfma_f32_16x16x32_f16(__builtin_bit_cast(f16x8, x), __builtin_bit_cast(f16x8, w), c, 0, 0, 0)

__global__ void __launch_bounds__(384, 2) a2a_gemm(Params p) {
    __shared__ __attribute__((aligned(16))) char smem[LDS_BYTES];
    const int b = blockIdx.x;
    u32x8 k0; u32x2 cfg; u64 dwsa, tr0;
    const auto ka = __builtin_amdgcn_kernarg_segment_ptr();
    asm volatile("s_load_dwordx8 %0, %4, 0\n\ts_load_dwordx2 %2, %4, %5 offset:0x68\n\ts_load_dwordx2 %1, %4, %6 offset:0x20\n\ts_memrealtime %3"
                 : "=&s"(k0), "=&s"(dwsa), "=&s"(cfg), "=&s"(tr0) : "s"(ka), "s"((b & 255) * 8), "s"((b & 7) * 8));
    SB();
    const int lane = threadIdx.x & 63;
    const int wave = __builtin_amdgcn_readfirstlane(threadIdx.x >> 6);
    LDSP char* lds = (LDSP char*)smem;
    volatile LDSP u32* sync = (volatile LDSP u32*)(lds + LDS_S);
#define ARGS() asm volatile("s_waitcnt lgkmcnt(0)" : "+s"(k0), "+s"(dwsa), "+s"(cfg), "+s"(tr0))
#define KPTR(i) ((char*)((u64)k0[2 * (i) + 1] << 32 | k0[2 * (i)]))

    if (__builtin_expect(wave == 5, 1)) {
        sync[lane & 15] = 0;
        ARGS();
        const u32 wo = DATA_OFF + ((cfg[1] >> 12) & 63) * 16384 + ((cfg[1] >> 6) & 7) * 2048;
        u32 w0; u64 tc0;
        asm volatile("s_load_dword %0, %2, %3\n\ts_load_dword %0, %2, %4\n\ts_load_dword %0, %2, %5\n\ts_memtime %1\n\ts_barrier\n\ts_waitcnt lgkmcnt(0)"
                     : "=&s"(w0), "=&s"(tc0) : "s"(dwsa), "s"(wo), "s"(wo + BUF_BYTES), "s"(wo + 2 * BUF_BYTES) : "memory");
        sync[2] = (u32)(__builtin_amdgcn_s_memtime() - tc0) | 1u;
        __builtin_amdgcn_endpgm();
    }
    if (__builtin_expect(wave >= 1, 0)) {
        const int sw = wave - 1;
        const int mt = (b >> 3) & 3, kh = b & 1, n0 = ((b >> 1) & 3) * 512 + (b >> 5) * 64;
        u32 wv = (u32)((kh * 1024 + sw * 256 + 8 * (lane >> 4)) * NG + n0 + 4 * (lane & 15)) * 4;
        asm volatile("; compute" : "+v"(wv));
        ARGS();
        const u32 tstart = (u32)tr0;
        const u32 ord = cfg[0];
        const u32x4 rw = mkrsrc(KPTR(1), (u32)KG * NG * 4);
        const u32x4 rs = mkrsrc(KPTR(3), WS_USED);
        __builtin_amdgcn_s_setprio(1);
        SB();
        __builtin_amdgcn_s_barrier();
        if ((cfg[1] >> 30) & 1)
            while (__builtin_amdgcn_readfirstlane(sync[1]) == 0) {}
        SB();
        f32x4 t[8][8];
        u32x4 wf[8][4];
        u32x4 aP[8][2], aL[8][2];
        f32x4 acc[2][4];
        f32x2 own[4];
        u32x4 y[2];
        u32 ko[8], kp[8], pv = 0, avP = 0, avL = 0, pr_ = 0, call = 0;
        u32 bad = 0, pend = 0, polled = 0, spin = 0;
        bool defer = false, waited = false;
        u64 bm;
        const f32x4 zero4 = {0, 0, 0, 0};
        u32 ya = XCH_OFF + (b * 4 + sw) * 2048 + lane * 16;
        u32 ov = (u32)((mt * 64 + kh * 32 + (sw >> 1) * 16 + (lane >> 4) * 4 + (sw & 1) * 2) * NG + n0 + (lane & 15) * 4) * 4, ov2 = ov + NG * 4;
        LDSP char* pw1 = lds + LDS_P + ((sw ^ 1) * 4 + 1) * 2048 + lane * 8;
        LDSP char* pw2 = lds + LDS_P + ((sw ^ 2) * 4 + 2) * 2048 + lane * 8;
        LDSP char* pw3 = lds + LDS_P + ((sw ^ 3) * 4 + 3) * 2048 + lane * 8;
        LDSP char* prd = lds + LDS_P + sw * 8192 + lane * 8;
        volatile LDSP u32* fw = sync + 4 + sw;
        volatile LDSP u32* fr = sync + 4 + (lane & 3);

#define KB(pp) ((((ord >> (4 * (pp))) & 7) + 2 * sw) & 7)
#define WPAIR(pp, jj) asm volatile("buffer_load_dwordx4 %0, %2, %3, %4 offen\n\tbuffer_load_dwordx4 %1, %2, %3, %5 offen" \
            : "=&v"(t[pp][2 * (jj)]), "=&v"(t[pp][2 * (jj) + 1]) \
            : "v"(wv), "s"(rw), "s"((KB(pp) * 32 + 2 * (jj)) * (NG * 4)), "s"((KB(pp) * 32 + 2 * (jj) + 1) * (NG * 4)))
#define WWAIT(pp, jj, N) asm volatile("s_waitcnt vmcnt(" #N ")" : "+v"(t[pp][2 * (jj)]), "+v"(t[pp][2 * (jj) + 1]))
#define WCVT(pp, jj) do { SB(); \
            _Pragma("unroll") for (int c = 0; c < 4; ++c) wf[pp][c][jj] = pk(t[pp][2 * (jj)][c], t[pp][2 * (jj) + 1][c]); \
            asm volatile("" :: "v"(wf[pp][0][jj]), "v"(wf[pp][1][jj]), "v"(wf[pp][2][jj]), "v"(wf[pp][3][jj])); SB(); } while (0)
#define ALOAD(A, voff, q) asm volatile("buffer_load_dwordx4 %0, %2, %3, %4 offen sc0 sc1\n\tbuffer_load_dwordx4 %1, %2, %3, %5 offen sc0 sc1" \
            : "=&v"(A[0]), "=&v"(A[1]) : "v"(voff), "s"(rs), "s"(ko[q]), "s"(kp[q]))
#define RELOAD1(X, q, v) do { if ((q) < 8) \
            asm volatile("buffer_load_dwordx4 %0, %2, %3, %4 offen sc0 sc1\n\tbuffer_load_dwordx4 %1, %2, %3, %5 offen sc0 sc1" \
            : "+v"(X[(q) & 7][0]), "+v"(X[(q) & 7][1]) : "v"(v), "s"(rs), "s"(ko[(q) & 7]), "s"(kp[(q) & 7])); } while (0)
#define AWAIT(A, N) asm volatile("s_waitcnt vmcnt(" #N ")" : "+v"(A[0]), "+v"(A[1]))
#define AWAITC(A, N) asm volatile("s_cmp_lg_u32 %2, 0\n\ts_cbranch_scc1 1f\n\ts_waitcnt vmcnt(" #N ")\n1:" : "+v"(A[0]), "+v"(A[1]) : "s"(polled) : "scc")
#define WAIT0() asm volatile("s_waitcnt vmcnt(0)" ::: "memory")
#define TIE(A) asm volatile("" : "+v"(A[0]), "+v"(A[1]))
#define DEF(A) asm volatile("" : "=v"(A[0]), "=v"(A[1]))
#define CLR(A) do { A[0] = u32x4{0, 0, 0, 0}; A[1] = u32x4{0, 0, 0, 0}; TIE(A); } while (0)
#define PREP() do { _Pragma("unroll") for (int q = 0; q < 8; ++q) { ko[q] = KB(q) * 2048 + (sw >> 1) * 1024; kp[q] = ko[q] ^ 1024; } } while (0)
#define GETCALL() do { \
            do { call = __builtin_amdgcn_readfirstlane(sync[0]); } while (call == 0); \
            const u32 ab = DATA_OFF + (call % 3u) * BUF_BYTES + ((kh * 4 + sw) * 4 + mt) * 32768; \
            avP = ab + (1 - kh) * 16384 + (lane ^ (2 * (sw & 1))) * 16; avL = ab + kh * 16384 + (lane ^ (2 * (sw & 1))) * 16; \
            ya += (call & 1) << 21; \
            pv = ab + (1 - kh) * 16384 + KB(3) * 2048 + (lane & 1) * 1024 + 1020; \
            asm volatile("" : "+v"(avP), "+v"(avL), "+v"(pv), "+v"(ya), "+v"(ov), "+v"(ov2), "+v"(pw1), "+v"(pw2), "+v"(pw3), "+v"(prd), "+v"(fw), "+v"(fr)); } while (0)
#define PROBE0() asm volatile("buffer_load_dword %0, %1, %2, 0 offen sc0 sc1" : "=v"(pr_) : "v"(pv), "s"(rs))
#define PWAIT(N) asm volatile("s_waitcnt vmcnt(" #N ")" : "+v"(pr_))

#define ZMIN(A) min(min3u(min3u(A[0][0], A[0][1], A[0][2]), A[1][2], A[1][3]), min3u(A[0][3], A[1][0], A[1][1]))
#define ZERO(A) (__builtin_amdgcn_ballot_w64(ZMIN(A) == 0) != 0)
#define LATEQ() ((u32)__builtin_amdgcn_s_memrealtime() - tstart > DEADLINE_TICKS)
#define RW2(X, q, v0) asm volatile("buffer_load_dwordx4 %0, %4, %5, %6 offen sc0 sc1\n\tbuffer_load_dwordx4 %1, %4, %5, %7 offen sc0 sc1\n\t" \
                "buffer_load_dwordx4 %2, %4, %5, %8 offen sc0 sc1\n\tbuffer_load_dwordx4 %3, %4, %5, %9 offen sc0 sc1\n\ts_waitcnt vmcnt(4)" \
                : "+v"(X[q][0]), "+v"(X[q][1]), "+v"(X[(q) < 7 ? (q) + 1 : 6][0]), "+v"(X[(q) < 7 ? (q) + 1 : 6][1]) \
                : "v"(v0), "s"(rs), "s"(ko[q]), "s"(kp[q]), "s"(ko[(q) < 7 ? (q) + 1 : 6]), "s"(kp[(q) < 7 ? (q) + 1 : 6]))
#define POLL(X, q, v0) do { \
                bool again_; \
                do { \
                    RW2(X, q, v0); \
                    again_ = ZERO(X[q]); \
                    if (again_) { __builtin_amdgcn_s_sleep(2); \
                        TIE(X[q]); again_ = ZERO(X[q]); \
                        if (again_ && (++spin & 7u) == 0 && LATEQ()) { bad = 1; again_ = false; } } \
                } while (again_); \
                RELOAD1(X, (q) + 2, v0); RELOAD1(X, (q) + 3, v0); RELOAD1(X, (q) + 4, v0); RELOAD1(X, (q) + 5, v0); \
            } while (0)

#define MM(X, r, q, c) do { acc[r][c] = MFMA(X[q][r], wf[q][c], (q) == 0 ? zero4 : acc[r][c]); SB(); } while (0)
#define MA(X, r, q, c) do { acc[r][c] = MFMA(X[q][r], wf[q][c], acc[r][c]); SB(); } while (0)
#define STEP8(X, q) do { MA(X, 0, q, 0); MA(X, 0, q, 1); MA(X, 0, q, 2); MA(X, 0, q, 3); MA(X, 1, q, 0); MA(X, 1, q, 1); MA(X, 1, q, 2); MA(X, 1, q, 3); } while (0)
#define PSTEP(q) do { SB(); if (__builtin_expect(ZERO(aP[q]), 0)) { pend |= 1u << (q); CLR(aP[q]); } \
            MM(aP, 0, q, 0); MM(aP, 0, q, 1); MM(aP, 0, q, 2); MM(aP, 0, q, 3); MM(aP, 1, q, 0); MM(aP, 1, q, 1); MM(aP, 1, q, 2); MM(aP, 1, q, 3); } while (0)
#define PCHK7() do { if (ZERO(aL[7])) pend |= 128u; } while (0)
#define REQ(q) do { if (pend >> (q) & 1) RELOAD1(aL, q, avP); } while (0)
#define HERE(q) do { TIE(aL[q]); if (ZERO(aL[q])) POLL(aL, q, avP); } while (0)
#define REDO(q) do { if (pend >> (q) & 1) { HERE(q); STEP8(aL, q); } } while (0)
#define CATCHUP(RL) do { \
            waited = true; \
            REQ(0); REQ(1); REQ(2); REQ(3); REQ(4); REQ(5); REQ(6); REQ(7); \
            WAIT0(); \
            REDO(0); REDO(1); REDO(2); REDO(3); REDO(4); REDO(5); REDO(6); HERE(7); \
            RL; } while (0)
#define FBODY(X, q, WAIT) do { \
            u32 m0_, m1_; \
            MM(X, 0, q, 0); MM(X, 0, q, 1); \
            WAIT; SB(); \
            MM(X, 0, q, 2); \
            m0_ = min3u(X[(q) + 1][0][0], X[(q) + 1][0][1], X[(q) + 1][0][2]); KEEP(m0_); \
            m1_ = min3u(X[(q) + 1][0][3], X[(q) + 1][1][0], X[(q) + 1][1][1]); KEEP(m1_); SB(); \
            MM(X, 0, q, 3); \
            m0_ = min3u(m0_, X[(q) + 1][1][2], X[(q) + 1][1][3]); KEEP(m0_); SB(); \
            MM(X, 1, q, 0); \
            m0_ = min(m0_, m1_); KEEP(m0_); SB(); \
            MM(X, 1, q, 1); \
            bm = __builtin_amdgcn_ballot_w64(m0_ == 0); asm volatile("" :: "s"(bm)); SB(); \
            MM(X, 1, q, 2); MM(X, 1, q, 3); \
            } while (0)
#define SST(X, q, v0) FBODY(X, q, TIE(X[(q) + 1])); \
            if (__builtin_expect(bm != 0, 0)) { POLL(X, (q) + 1, v0); }
#define CST(X, q, N, v0) FBODY(X, q, AWAITC(X[(q) + 1], N)); \
            if (__builtin_expect(bm != 0, 0)) { POLL(X, (q) + 1, v0); polled = 1; }

#define LO(v) f32x2{(v)[0], (v)[1]}
#define HI(v) f32x2{(v)[2], (v)[3]}
#define PW2(cls, c) do { *(LDSP f32x2*)(pw2 + (cls) * 32768 + (c) * 512) = LO(acc[1][c]); *(LDSP f32x2*)(pw3 + (cls) * 32768 + (c) * 512) = HI(acc[1][c]); SB(); } while (0)
#define PW1(cls, c) do { *(LDSP f32x2*)(pw1 + (cls) * 32768 + (c) * 512) = HI(acc[0][c]); SB(); } while (0)
#define LASTW(X, cls, F0, F1, F2, F3, F4, F5) do { \
            if (__builtin_expect(bad, 0)) { const float nan = __builtin_nanf(""); acc[0][0] = f32x4{nan, nan, nan, nan}; acc[1][0] = acc[0][0]; } \
            MM(X, 1, 7, 0); MM(X, 1, 7, 1); F0; MM(X, 1, 7, 2); F1; MM(X, 1, 7, 3); PW2(cls, 0); MM(X, 0, 7, 0); PW2(cls, 1); MM(X, 0, 7, 1); \
            PW2(cls, 2); MM(X, 0, 7, 2); PW2(cls, 3); MM(X, 0, 7, 3); PW1(cls, 0); PW1(cls, 1); PW1(cls, 2); PW1(cls, 3); \
            asm volatile("" ::: "memory"); fw[4 * (cls)] = 1; SB(); F2; F3; F4; F5; SB(); } while (0)
#define TRYG(cls, OK) do { \
            asm volatile("" ::: "memory"); \
            const u32 fl_ = fr[4 * (cls)]; \
            asm volatile("" ::: "memory"); SB(); \
            _Pragma("unroll") for (int k = 0; k < 3; ++k) _Pragma("unroll") for (int c = 0; c < 4; ++c) \
                q_[c][k] = *(LDSP f32x2*)(prd + (cls) * 32768 + (k + 1) * 2048 + c * 512); \
            SB(); \
            OK = __builtin_amdgcn_ballot_w64(fl_ == 0) == 0; } while (0)
#define FADD(d, x, y_) asm("v_add_f32 %0, %1, %2" : "=v"(d) : "v"(x), "v"(y_))
#define SEND() do { \
            u32x4 e0_, e1_; \
            _Pragma("unroll") for (int c = 0; c < 4; ++c) { \
                const u32x2 z_ = __builtin_bit_cast(u32x2, (own[c] + q_[c][0]) + (q_[c][1] + q_[c][2])); \
                e0_[c] = NZ(z_[0]); e1_[c] = NZ(z_[1]); } \
            asm volatile("buffer_store_dwordx4 %0, %2, %3, 0 offen sc1\n\tbuffer_store_dwordx4 %1, %2, %3, 0 offen offset:1024 sc1\n\ts_nop 1" \
                         :: "v"(e0_), "v"(e1_), "v"(ya ^ 8192u), "s"(rs) : "memory"); } while (0)
#define HANDOFF(X, PRE) do { \
            _Pragma("unroll") for (int c = 0; c < 4; ++c) own[c] = LO(acc[0][c]); \
            f32x2 q_[4][3]; bool ok_; int n_ = 0; \
            do { TRYG(0, ok_); } while (!ok_ && ++n_ < 8); \
            defer = !ok_; \
            PRE; \
            if (__builtin_expect(ok_, 1)) SEND(); else { WAIT0(); polled = 1; } } while (0)
#define RETRY() do { if (__builtin_expect(defer, 0)) { f32x2 q_[4][3]; bool ok_; TRYG(0, ok_); if (ok_) { SEND(); defer = false; } } } while (0)
#define YRELOAD() asm volatile("buffer_load_dwordx4 %0, %2, %3, 0 offen sc1\n\tbuffer_load_dwordx4 %1, %2, %3, 0 offen offset:1024 sc1" \
                         : "+v"(y[0]), "+v"(y[1]) : "v"(ya), "s"(rs))
#define LCLASS(X, v0) \
            AWAITC(X[0], 16); if (__builtin_expect(ZERO(X[0]), 0)) { POLL(X, 0, v0); waited = true; polled = 1; } \
            CST(X, 0, 14, v0) CST(X, 1, 12, v0) RETRY(); CST(X, 2, 10, v0) CST(X, 3, 8, v0) RETRY(); \
            AWAITC(X[7], 2); polled = 1; \
            if (__builtin_expect(waited, 0)) { if (ZERO(X[7])) { RELOAD1(X, 5, v0); RELOAD1(X, 6, v0); RELOAD1(X, 7, v0); } } \
            CLR(y); YRELOAD(); \
            SST(X, 4, v0) SST(X, 5, v0) RETRY(); SST(X, 6, v0) YRELOAD();
#define TAIL(X) do { \
            f32x2 q_[4][3]; bool ok_; \
            if (__builtin_expect(defer, 0)) { do { TRYG(0, ok_); } while (!ok_); SEND(); } \
            TRYG(1, ok_); \
            float s0_[4], s1_[4]; \
            _Pragma("unroll") for (int c = 0; c < 4; ++c) { s0_[c] = acc[0][c][0]; s1_[c] = acc[0][c][1]; } \
            TIE(y); \
            bool ybad_ = ZERO(y); \
            while (__builtin_expect(ybad_, 0)) { \
                YRELOAD(); AWAIT(y, 2); \
                ybad_ = ZERO(y); \
                if (ybad_ && (++spin & 15u) == 0 && LATEQ()) { bad = 1; ybad_ = false; } \
            } \
            _Pragma("unroll") for (int c = 0; c < 4; ++c) { FADD(s0_[c], s0_[c], y[0][c]); FADD(s1_[c], s1_[c], y[1][c]); } \
            while (!ok_) TRYG(1, ok_); \
            if (__builtin_expect(bad, 0)) s0_[0] = __builtin_nanf(""); \
            _Pragma("unroll") for (int c = 0; c < 4; ++c) { const f32x2 v_ = (q_[c][0] + q_[c][1]) + q_[c][2]; FADD(s0_[c], s0_[c], v_[0]); FADD(s1_[c], s1_[c], v_[1]); } \
            auto ro = __builtin_amdgcn_make_buffer_rsrc((void*)KPTR(2), 0, MS * NG * 4, 0x00020000); \
            const f32x4 o0 = gelu4(f32x4{s0_[0], s0_[1], s0_[2], s0_[3]}); \
            __builtin_amdgcn_raw_buffer_store_b128(__builtin_bit_cast(u32x4, o0), ro, ov, 0, 17); \
            SB(); const f32x4 o1 = gelu4(f32x4{s1_[0], s1_[1], s1_[2], s1_[3]}); \
            __builtin_amdgcn_raw_buffer_store_b128(__builtin_bit_cast(u32x4, o1), ro, ov2, 0, 17); \
              \
            asm volatile("" :: "v"(X[0][0]), "v"(X[0][1]), "v"(X[1][0]), "v"(X[1][1]), "v"(X[2][0]), "v"(X[2][1]), "v"(X[3][0]), "v"(X[3][1]), \
                "v"(X[4][0]), "v"(X[4][1]), "v"(X[5][0]), "v"(X[5][1]), "v"(X[6][0]), "v"(X[6][1]), "v"(X[7][0]), "v"(X[7][1]), "v"(y[0]), "v"(y[1])); \
            __builtin_amdgcn_endpgm(); } while (0)

        WPAIR(0, 0); WPAIR(0, 1); WPAIR(0, 2); WPAIR(0, 3); PREP(); WWAIT(0, 0, 6); WPAIR(1, 0); WCVT(0, 0); WWAIT(0, 1, 6); WPAIR(1, 1);
        WCVT(0, 1); WWAIT(0, 2, 6); WPAIR(1, 2); WCVT(0, 2); WWAIT(0, 3, 6); WPAIR(1, 3); WCVT(0, 3); WWAIT(1, 0, 6); WPAIR(2, 0);
        WCVT(1, 0); WWAIT(1, 1, 6); WPAIR(2, 1); WCVT(1, 1); WWAIT(1, 2, 6); WPAIR(2, 2); WCVT(1, 2); WWAIT(1, 3, 6); WPAIR(2, 3);
        WCVT(1, 3); WWAIT(2, 0, 6); WPAIR(3, 0); WCVT(2, 0); WWAIT(2, 1, 6); WPAIR(3, 1); WCVT(2, 1); WWAIT(2, 2, 6); WPAIR(3, 2);
        WCVT(2, 2); WWAIT(2, 3, 6); WPAIR(3, 3); WCVT(2, 3); WWAIT(3, 0, 6); WPAIR(4, 0); WCVT(3, 0); WWAIT(3, 1, 6); WPAIR(4, 1);
        WCVT(3, 1); WWAIT(3, 2, 6); WPAIR(4, 2); WCVT(3, 2); WWAIT(3, 3, 6); WPAIR(4, 3); WCVT(3, 3); WWAIT(4, 0, 6); GETCALL(); PROBE0();
        WPAIR(5, 0); WCVT(4, 0); WWAIT(4, 1, 7); WPAIR(5, 1); WCVT(4, 1); WWAIT(4, 2, 7); WPAIR(5, 2); WCVT(4, 2); WWAIT(4, 3, 7);
        WPAIR(5, 3); WCVT(4, 3); PWAIT(8);
        if (__builtin_expect(__builtin_amdgcn_ballot_w64(pr_ == 0) == 0, 1)) {
            WWAIT(5, 0, 6); ALOAD(aP[0], avP, 0); WPAIR(6, 0); WCVT(5, 0); WWAIT(5, 1, 8); ALOAD(aP[1], avP, 1); WCVT(5, 1); WWAIT(5, 2, 8);
            WPAIR(6, 1); WCVT(5, 2); WWAIT(5, 3, 8); ALOAD(aP[2], avP, 2); WCVT(5, 3); AWAIT(aP[0], 8); WPAIR(6, 2); PSTEP(0); WWAIT(6, 0, 8);
            ALOAD(aP[3], avP, 3); WCVT(6, 0); AWAIT(aP[1], 8); WPAIR(6, 3); PSTEP(1); WWAIT(6, 1, 8); ALOAD(aP[4], avP, 4); WCVT(6, 1);
            AWAIT(aP[2], 8); WPAIR(7, 0); PSTEP(2); WWAIT(6, 2, 8); ALOAD(aP[5], avP, 5); WCVT(6, 2); AWAIT(aP[3], 8); WPAIR(7, 1); PSTEP(3);
            WWAIT(6, 3, 8); ALOAD(aP[6], avP, 6); WCVT(6, 3); AWAIT(aP[4], 8); WPAIR(7, 2); PSTEP(4); WWAIT(7, 0, 8); ALOAD(aL[7], avP, 7);
            WCVT(7, 0); AWAIT(aP[5], 8); WPAIR(7, 3); PSTEP(5); WWAIT(7, 1, 8); ALOAD(aL[0], avL, 0); WCVT(7, 1); AWAIT(aP[6], 8);
            ALOAD(aL[1], avL, 1); PSTEP(6); WWAIT(7, 2, 8); ALOAD(aL[2], avL, 2); WCVT(7, 2); AWAIT(aL[7], 8); PCHK7();
            WWAIT(7, 3, 6); WCVT(7, 3); DEF(aL[3]); DEF(aL[4]); DEF(aL[5]); DEF(aL[6]);
            if (__builtin_expect(pend != 0, 0)) CATCHUP(RELOAD1(aL, 0, avL); RELOAD1(aL, 1, avL); RELOAD1(aL, 2, avL));
            LASTW(aL, 0, RELOAD1(aL, 3, avL), RELOAD1(aL, 4, avL), RELOAD1(aL, 5, avL), RELOAD1(aL, 6, avL), , RELOAD1(aL, 7, avL));
            HANDOFF(aL, ); LCLASS(aL, avL) LASTW(aL, 1, , , , , , ); TAIL(aL);
        } else {
            WWAIT(5, 0, 6); WPAIR(6, 0); WCVT(5, 0); WWAIT(5, 1, 6); WPAIR(6, 1); WCVT(5, 1); WWAIT(5, 2, 6); WPAIR(6, 2); WCVT(5, 2);
            WWAIT(5, 3, 6); WPAIR(6, 3); WCVT(5, 3); WWAIT(6, 0, 6); WPAIR(7, 0); WCVT(6, 0); WWAIT(6, 1, 6); WPAIR(7, 1); WCVT(6, 1);
            WWAIT(6, 2, 6); WPAIR(7, 2); WCVT(6, 2); WWAIT(6, 3, 6); WPAIR(7, 3); WCVT(6, 3); WWAIT(7, 0, 6); ALOAD(aL[0], avP, 0);
            WCVT(7, 0); WWAIT(7, 1, 6); ALOAD(aL[1], avP, 1); WCVT(7, 1); WWAIT(7, 2, 6); ALOAD(aL[2], avP, 2); WCVT(7, 2); WWAIT(7, 3, 6);
            ALOAD(aL[3], avP, 3); WCVT(7, 3); ALOAD(aL[4], avP, 4); ALOAD(aL[5], avP, 5); ALOAD(aL[6], avP, 6); ALOAD(aL[7], avP, 7);
            waited = true; AWAIT(aL[0], 14); if (__builtin_expect(ZERO(aL[0]), 0)) { POLL(aL, 0, avP); polled = 1; } CST(aL, 0, 12, avP)
            CST(aL, 1, 10, avP) CST(aL, 2, 8, avP) CST(aL, 3, 6, avP) CST(aL, 4, 4, avP) CST(aL, 5, 2, avP) CST(aL, 6, 0, avP) polled = 0;
            LASTW(aL, 0, RELOAD1(aL, 0, avL), RELOAD1(aL, 1, avL), RELOAD1(aL, 2, avL); RELOAD1(aL, 3, avL), , , );
            HANDOFF(aL, RELOAD1(aL, 4, avL); RELOAD1(aL, 5, avL); RELOAD1(aL, 6, avL); RELOAD1(aL, 7, avL)); LCLASS(aL, avL) LASTW(aL, 1, , , , , , ); TAIL(aL);
        }
    }

    {
        u32 xv = (u32)((lane >> 4) * KS + (lane & 7) * 4) * 4, xsel = 6 + (lane & 8) * 3 / 8;
        asm volatile("; sender" : "+v"(xv), "+v"(xsel));
        ARGS();
        const u32 snd = cfg[1];
        xv += ((snd >> xsel) & 7) * 128;
        __builtin_amdgcn_s_setprio(3);
        auto rx = __builtin_amdgcn_make_buffer_rsrc((void*)KPTR(0), 0, MG * KS * 4, 0x00020000);
        auto rws = __builtin_amdgcn_make_buffer_rsrc((void*)KPTR(3), 0, WS_USED, 0x00020000);
        const u32 xs = (snd & 63) << 15;
        f32x4 v[8];
        SB();
        u32 cv = __builtin_amdgcn_raw_buffer_load_b32(rws, 0, CNT_OFF + b * 128, 16);
#pragma unroll
        for (int j = 0; j < 8; ++j) v[j] = __builtin_bit_cast(f32x4, __builtin_amdgcn_raw_buffer_load_b128(rx, xv, xs + j * 4096, 0));
        SB();
        __builtin_amdgcn_s_barrier();
        SB();
        const u64 tc0 = __builtin_amdgcn_s_memtime();
        const int k4 = lane & 15;
        LDSP char* xw = lds + LDS_X + (k4 >> 3) * 2240 + ((k4 >> 1) & 3) * 272 + (lane >> 4) * 16 + (k4 & 1) * 8;
        LDSP char* xr = lds + LDS_X + (lane >> 4) * 272 + (lane & 15) * 16;
        u32x4 zz = {0, 0, 0, 0};
        u32 tm = ((snd >> 21) & 511) * 16;
        asm volatile("" : "+v"(xw), "+v"(xr), "+v"(zz), "+s"(tm));
        auto rd = __builtin_amdgcn_make_buffer_rsrc((void*)dwsa, 0, WS_USED, 0x00020000);
        SB();
        asm volatile("" : "+v"(cv));
        const u32 call = __builtin_amdgcn_readfirstlane(cv) + 1;
        const u32 c3 = call % 3u;
        const u32 z3 = c3 == 0 ? 2u : c3 - 1;
        sync[0] = call;
        u32 dv = lane * 16 + DATA_OFF + c3 * BUF_BYTES + ((snd >> 12) & 63) * 16384 + ((snd >> 6) & 7) * 2048;
        u32 dv2 = lane * 16 + DATA_OFF + c3 * BUF_BYTES + ((snd >> 12) & 63) * 16384 + ((snd >> 9) & 7) * 2048;
        u32 zv = lane * 16 + DATA_OFF + z3 * BUF_BYTES + b * 4096;
        u32 zx = XCH_OFF + ((~call & 1) << 21) + b * 8192 + lane * 16, zx2 = zx + 4096;
        asm volatile("" : "+v"(dv), "+v"(dv2), "+v"(zv), "+v"(zx), "+v"(zx2));
        SB();
        u32 mn = 0xFFFFFFFFu;
#pragma unroll
        for (int j = 0; j < 8; ++j) {
            const u32x2 h = {pk(v[j][0], v[j][1]), pk(v[j][2], v[j][3])};
            *(LDSP u32x2*)(xw + (j >> 2) * 1120 + (j & 3) * 64) = h;
            mn = min3u(h[0], h[1], mn);
        }
        u32x4 hh[4];
#pragma unroll
        for (int f = 0; f < 4; ++f) hh[f] = *(LDSP u32x4*)(xr + f * 1120);
        SB();
        if (__builtin_expect(__builtin_amdgcn_ballot_w64(mn == 0) != 0, 0)) {
#pragma unroll
            for (int f = 0; f < 4; ++f)
#pragma unroll
                for (int e = 0; e < 4; ++e) hh[f][e] = NZ(hh[f][e]);
        }
        if (tm != 0) {
            if (tm >= 2300) {
                u32 tr;
                do { tr = __builtin_amdgcn_readfirstlane(sync[2]); } while (tr == 0);
                tm += min(tr - min(tr, 2600u), 20000u);
            }
            while ((u32)__builtin_amdgcn_s_memtime() - (u32)tc0 < tm) {}
        }
        SB();
#pragma unroll
        for (int f = 0; f < 4; ++f) __builtin_amdgcn_raw_buffer_store_b128(hh[f], rd, (f < 2 ? dv : dv2) + (f & 1) * 1024, 0, 17);
        sync[1] = 1;
#pragma unroll
        for (int f = 0; f < 4; ++f) __builtin_amdgcn_raw_buffer_store_b128(zz, rws, zv + f * 1024, 0, 17);
#pragma unroll
        for (int f = 0; f < 8; ++f) __builtin_amdgcn_raw_buffer_store_b128(zz, rws, (f < 4 ? zx : zx2) + (f & 3) * 1024, 0, 16);
        __builtin_amdgcn_raw_buffer_store_b32(call, rws, 0, CNT_OFF + b * 128, 16);
        asm volatile("s_waitcnt vmcnt(0)" ::: "memory");
    }
}

extern "C" void kernel_launch(int rank, int nranks, ncclComm_t comm,
                              hipStream_t stream,
                              void* const* d_in, const int* in_sizes, int n_in,
                              void* d_out, int out_size,
                              const int* in_dtypes, int out_dtype,
                              void* const* d_out_all,
                              void* const* d_ws_all, size_t ws_bytes) {
    if (nranks != NR || (unsigned)rank >= (unsigned)NR || n_in < 2 || ws_bytes < WS_USED ||
        in_sizes[0] != MG * KS || in_sizes[1] != KG * NG || out_size != MS * NG) {
        fprintf(stderr, "kernel_launch: unexpected problem shape\n");
        return;
    }
    Params p;
    p.x = (const float*)d_in[0];
    p.w = (const float*)d_in[1];
    p.out = (float*)d_out;
    p.myws = (char*)d_ws_all[rank];
    for (int i = 0; i < NR; ++i) p.ws[i] = (char*)d_ws_all[(rank + 1 + i) & 7];
    p.pad = 0;
    static const u32 ord[8] = {0x74561320, 0x65470231, 0x76532410, 0x76452301,
                               0x45761320, 0x65470231, 0x76543102, 0x74652013};
    static const char slots[33] = "30142032103633727241031025150261";
    const unsigned hold = 0x27F;
    for (int x = 0; x < 8; ++x) {
        int role[32], who[32];
        bool used[32] = {};
        for (int i = 0; i < 32; ++i) {
            const int tile = (x >> 1) * 8 + (i >> 2);
            bool slow = false;
            for (int l = 0; l < 2; ++l) {
                const unsigned line = (unsigned)(((size_t)d_in[1] >> 7) + 2 * tile + l);
                if ((line & 7) == 3 || (line & 63) == 63) slow = true;
            }
            role[i] = (i & 3) + (slow ? 4 : 0);
        }
        for (int pass = 0; pass < 3; ++pass)
            for (int sg = 0; sg < 32; ++sg) {
                if (pass == 0) who[sg] = -1;
                if (who[sg] >= 0) continue;
                const int want = slots[sg] - '0';
                for (int i = 0; i < 32; ++i)
                    if (!used[i] && (pass == 2 || role[i] == want || (pass == 1 && (role[i] & 3) == (want & 3)))) { who[sg] = i; used[i] = true; break; }
            }
        for (int sg = 0; sg < 32; ++sg) {
            const int b = who[sg] * 8 + x;
            const int umt = sg & 3, u = (sg >> 2) & 3, dest = (rank + 1 + x) & 7, half = ((sg >> 4) ^ 1 ^ (rank >> 2)) & 1;
            const u32 kA = ((ord[umt] >> (8 * u)) + 2 * rank) & 7, kB = ((ord[umt] >> (8 * u + 4)) + 2 * rank) & 7;
            const int tt = sg < 16 ? 1300 + sg * 140 : 3700 + (sg - 16) * 151;
            const u32 tm = x == 7 ? 0 : (u32)tt / 16;
            p.cfg[b][0] = ord[role[who[sg]]];
            p.cfg[b][1] = (u32)(dest * 8 + umt * 2 + half) | kA << 6 | kB << 9 | (u32)((rank * 4 + umt) * 2 + half) << 12
                          | tm << 21 | ((hold >> sg) & 1) << 30;
        }
    }
    a2a_gemm<<<dim3(256), dim3(384), 0, stream>>>(p);
    HIP_CHECK(hipGetLastError());
}
```
